```python
import numpy as np

kernel_name = "gpu_a2a_gemm_m2048_k2048_n2048_bf16_relu"


ABSMAX_RTOL = 0.008

COMPUTE_DTYPE = "bfloat16"
EPILOGUE = "relu"
IN_QUANT = None
OUT_QUANT = None
IN_DTYPES = {"x": "bfloat16", "w_mat": "bfloat16"}
WORLD_SIZE = 8

def _to_bf16(x):
    u = np.ascontiguousarray(x, dtype=np.float32).view(np.uint32)
    return ((u + 0x7FFF + ((u >> 16) & 1)) & 0xFFFF0000).view(np.float32)

def setup_inputs():

    x = _to_bf16(np.random.randn(2048, 2048).astype(np.float32))
    w_mat = _to_bf16((np.random.randn(2048, 2048) / np.sqrt(2048)).astype(np.float32))
    return {"x": x, "w_mat": w_mat}

def _epilogue(y):
    return np.maximum(y, 0.0).astype(np.float32)

def reference(x, w_mat):
    y = (x @ w_mat).astype(np.float32)
    return _epilogue(y)
```

```cpp
#include <hip/hip_runtime.h>
#include <rccl/rccl.h>
#include <cstdio>
#include <cstdint>

typedef __attribute__((ext_vector_type(8))) short bf16x8;
typedef __attribute__((ext_vector_type(16))) float f32x16;
typedef __attribute__((ext_vector_type(4))) unsigned int u32x4;
typedef __attribute__((ext_vector_type(2))) float f32x2;

#define NRANKS 8
#define NWG 256
#define NTHR 512
#define DEADLINE_TICKS 50000000ull

#define WS_CTR 0
#define WS_TMO 128
#define WS_FLAGS 4096
#define WS_STAGE (1u << 20)
#define SLOT_BYTES (1u << 20)

__device__ __forceinline__ unsigned int f2bf_pack(float lo, float hi) {
    unsigned short a = __builtin_bit_cast(unsigned short, (__bf16)lo);
    unsigned short b = __builtin_bit_cast(unsigned short, (__bf16)hi);
    return (unsigned int)a | ((unsigned int)b << 16);
}

__global__ void __launch_bounds__(NTHR, 2)
a2a_gemm_kernel(const unsigned short* __restrict__ x, const unsigned short* __restrict__ w,
                unsigned short* __restrict__ out, void* const* __restrict__ ws_all, int rank) {
    __shared__ __attribute__((aligned(16))) float red[8 * 2 * 16 * 64];
    __shared__ unsigned int sh_call;
    const int tid = threadIdx.x;
    const int lane = tid & 63;
    const int wave = tid >> 6;
    const int b = blockIdx.x;
    char* myws = (char*)ws_all[rank];

    unsigned int ctr = __hip_atomic_load((unsigned int*)(myws + WS_CTR), __ATOMIC_RELAXED, __HIP_MEMORY_SCOPE_AGENT);

    u32x4 xv;
    const int pd = (rank + (b & 7)) & 7;
    const int prb = b >> 5, pq = (b >> 3) & 3;
    if (tid < 256) {
        const int ks = tid >> 6, l = tid & 63;
        const int row = pd * 256 + prb * 32 + (l & 31);
        const int col = pq * 64 + ks * 16 + (l >> 5) * 8;
        xv = *(const u32x4*)(x + (size_t)row * 256 + col);
    }
    const unsigned int call = ctr / NWG + 1;
    if (tid < 256) {
        char* dws = (char*)ws_all[pd];
        u32x4* dst = (u32x4*)(dws + WS_STAGE + (call & 1) * SLOT_BYTES + ((rank * 8 + prb) * 16 + pq * 4) * 1024) + tid;
        *dst = xv;
    }

    const int idx = b >> 3;
    const int rb = idx >> 2;
    const int cb = (b & 7) * 4 + (idx & 3);
    const int n0 = cb * 64;

    unsigned int wr[16][8];
    {
        const unsigned short* wp = w + (size_t)(wave * 256 + (lane >> 5) * 8) * 2048 + n0 + 2 * (lane & 31);
#pragma unroll
        for (int ks = 0; ks < 16; ++ks)
#pragma unroll
            for (int j = 0; j < 8; ++j)
                wr[ks][j] = *(const unsigned int*)(wp + (size_t)(ks * 16 + j) * 2048);
    }

    asm volatile("s_waitcnt vmcnt(0)" ::: "memory");
    __syncthreads();
    if (tid == 0) {
        __builtin_amdgcn_fence(__ATOMIC_RELEASE, "");
        asm volatile("s_waitcnt vmcnt(0)" ::: "memory");
        char* dws = (char*)ws_all[pd];
        __hip_atomic_store((unsigned int*)(dws + WS_FLAGS + (rank * 8 + prb) * 128 + pq * 4), call,
                           __ATOMIC_RELAXED, __HIP_MEMORY_SCOPE_SYSTEM);
    }

    unsigned int* tmo = (unsigned int*)(myws + WS_TMO);
    {
        const unsigned int* fl = (const unsigned int*)(myws + WS_FLAGS + (wave * 8 + rb) * 128) + (lane & 3);
        unsigned long long t0 = __builtin_amdgcn_s_memrealtime();
        while (true) {
            unsigned int v = __hip_atomic_load(fl, __ATOMIC_RELAXED, __HIP_MEMORY_SCOPE_SYSTEM);
            if (__all(v >= call)) break;
            if (__hip_atomic_load(tmo, __ATOMIC_RELAXED, __HIP_MEMORY_SCOPE_AGENT) == call
                || __builtin_amdgcn_s_memrealtime() - t0 > DEADLINE_TICKS) {
                if (lane == 0) __hip_atomic_store(tmo, call, __ATOMIC_RELAXED, __HIP_MEMORY_SCOPE_AGENT);
                break;
            }
            __builtin_amdgcn_s_sleep(1);
        }
        __builtin_amdgcn_fence(__ATOMIC_ACQUIRE, "");
    }

    f32x16 acc0 = {0}, acc1 = {0};
    {
        const u32x4* ap = (const u32x4*)(myws + WS_STAGE + (call & 1) * SLOT_BYTES + (wave * 8 + rb) * 16 * 1024) + lane;
        u32x4 a[16];
#pragma unroll
        for (int ks = 0; ks < 16; ++ks) a[ks] = ap[ks * 64];
#pragma unroll
        for (int ks = 0; ks < 16; ++ks) {
            u32x4 b0, b1;
#pragma unroll
            for (int j = 0; j < 4; ++j) {
                b0[j] = __builtin_amdgcn_perm(wr[ks][2 * j + 1], wr[ks][2 * j], 0x05040100u);
                b1[j] = __builtin_amdgcn_perm(wr[ks][2 * j + 1], wr[ks][2 * j], 0x07060302u);
            }
            acc0 = __builtin_amdgcn_mfma_f32_32x32x16_bf16(__builtin_bit_cast(bf16x8, a[ks]), __builtin_bit_cast(bf16x8, b0), acc0, 0, 0, 0);
            acc1 = __builtin_amdgcn_mfma_f32_32x32x16_bf16(__builtin_bit_cast(bf16x8, a[ks]), __builtin_bit_cast(bf16x8, b1), acc1, 0, 0, 0);
        }
    }

#pragma unroll
    for (int i = 0; i < 8; ++i) {
        *(f32x2*)&red[(((wave * 2 + 0) * 8 + i) * 64 + lane) * 2] = (f32x2){acc0[2 * i], acc0[2 * i + 1]};
        *(f32x2*)&red[(((wave * 2 + 1) * 8 + i) * 64 + lane) * 2] = (f32x2){acc1[2 * i], acc1[2 * i + 1]};
    }
    __syncthreads();
    const bool bad = __hip_atomic_load(tmo, __ATOMIC_RELAXED, __HIP_MEMORY_SCOPE_AGENT) == call;
    {
        f32x2 s0 = {0, 0}, s1 = {0, 0};
#pragma unroll
        for (int v = 0; v < 8; ++v) {
            s0 += *(const f32x2*)&red[(((v * 2 + 0) * 8 + wave) * 64 + lane) * 2];
            s1 += *(const f32x2*)&red[(((v * 2 + 1) * 8 + wave) * 64 + lane) * 2];
        }
#pragma unroll
        for (int e = 0; e < 2; ++e) {
            const int i = 2 * wave + e;
            const int row = rb * 32 + (i & 3) + 8 * (i >> 2) + 4 * (lane >> 5);
            float v0 = fmaxf(s0[e], 0.f), v1 = fmaxf(s1[e], 0.f);
            unsigned int pk = f2bf_pack(v0, v1);
            if (bad) pk = 0x7fc07fc0u;
            *(unsigned int*)(out + (size_t)row * 2048 + n0 + 2 * (lane & 31)) = pk;
        }
    }
    if (tid == 0)
        __hip_atomic_fetch_add((unsigned int*)(myws + WS_CTR), 1u, __ATOMIC_RELAXED, __HIP_MEMORY_SCOPE_AGENT);
    (void)sh_call;
}

extern "C" void kernel_launch(int rank, int nranks, ncclComm_t comm,
                              hipStream_t stream,
                              void* const* d_in, const int* in_sizes, int n_in,
                              void* d_out, int out_size,
                              const int* in_dtypes, int out_dtype,
                              void* const* d_out_all,
                              void* const* d_ws_all, size_t ws_bytes) {
    if (nranks != NRANKS || (unsigned)rank >= NRANKS) return;
    hipLaunchKernelGGL(a2a_gemm_kernel, dim3(NWG), dim3(NTHR), 0, stream,
                       (const unsigned short*)d_in[0], (const unsigned short*)d_in[1],
                       (unsigned short*)d_out, d_ws_all, rank);
}
```

```cpp
#include <hip/hip_runtime.h>
#include <rccl/rccl.h>
#include <cstdio>
#include <cstdint>

typedef __attribute__((ext_vector_type(8))) short bf16x8;
typedef __attribute__((ext_vector_type(16))) float f32x16;
typedef __attribute__((ext_vector_type(4))) unsigned int u32x4;
typedef __attribute__((ext_vector_type(2))) float f32x2;

#define NRANKS 8
#define NWG 256
#define NTHR 512
#define DEADLINE_TICKS 50000000ull

#define WS_CTR 0
#define WS_TMO 128
#define WS_FLAGS 4096
#define WS_STAGE (1u << 20)
#define SLOT_BYTES (1u << 20)

__device__ __forceinline__ unsigned int f2bf_pack(float lo, float hi) {
    unsigned short a = __builtin_bit_cast(unsigned short, (__bf16)lo);
    unsigned short b = __builtin_bit_cast(unsigned short, (__bf16)hi);
    return (unsigned int)a | ((unsigned int)b << 16);
}

#define STAMP(i) do { } while (0)

__global__ void __launch_bounds__(NTHR, 2)
a2a_gemm_kernel(const unsigned short* __restrict__ x, const unsigned short* __restrict__ w,
                unsigned short* __restrict__ out, void* const* __restrict__ ws_all, int rank) {
    __shared__ __attribute__((aligned(16))) float red[8 * 2 * 16 * 64];
    __shared__ unsigned int sh_call;
    const int tid = threadIdx.x;
    const int lane = tid & 63;
    const int wave = tid >> 6;
    const int b = blockIdx.x;
    char* myws = (char*)ws_all[rank];
    STAMP(0);

    unsigned int ctr = __hip_atomic_load((unsigned int*)(myws + WS_CTR), __ATOMIC_RELAXED, __HIP_MEMORY_SCOPE_AGENT);

    u32x4 xv;
    const int pd = (rank + (b & 7)) & 7;
    const int prb = b >> 5, pq = (b >> 3) & 3;
    if (tid < 256) {
        const int ks = tid >> 6, l = tid & 63;
        const int row = pd * 256 + prb * 32 + (l & 31);
        const int col = pq * 64 + ks * 16 + (l >> 5) * 8;
        xv = *(const u32x4*)(x + (size_t)row * 256 + col);
    }
    const unsigned int call = ctr / NWG + 1;
    if (tid < 256) {
        char* dws = (char*)ws_all[pd];
        u32x4* dst = (u32x4*)(dws + WS_STAGE + (call & 1) * SLOT_BYTES + ((rank * 8 + prb) * 16 + pq * 4) * 1024) + tid;
        unsigned long long* d8 = (unsigned long long*)dst;
        __hip_atomic_store(d8, (unsigned long long)xv[0] | ((unsigned long long)xv[1] << 32), __ATOMIC_RELAXED, __HIP_MEMORY_SCOPE_SYSTEM);
        __hip_atomic_store(d8 + 1, (unsigned long long)xv[2] | ((unsigned long long)xv[3] << 32), __ATOMIC_RELAXED, __HIP_MEMORY_SCOPE_SYSTEM);
    }
    asm volatile("s_waitcnt vmcnt(0)" ::: "memory");
    STAMP(1);
    __syncthreads();
    if (tid == 0) {
        char* dws = (char*)ws_all[pd];
        __hip_atomic_store((unsigned int*)(dws + WS_FLAGS + (rank * 8 + prb) * 128 + pq * 4), call,
                           __ATOMIC_RELAXED, __HIP_MEMORY_SCOPE_SYSTEM);
    }

    const int idx = b >> 3;
    const int rb = idx >> 2;
    const int cb = (b & 7) * 4 + (idx & 3);
    const int n0 = cb * 64;

    unsigned int wr[16][8];
    {
        const unsigned short* wp = w + (size_t)(wave * 256 + (lane >> 5) * 8) * 2048 + n0 + 2 * (lane & 31);
#pragma unroll
        for (int ks = 0; ks < 16; ++ks)
#pragma unroll
            for (int j = 0; j < 8; ++j)
                wr[ks][j] = *(const unsigned int*)(wp + (size_t)(ks * 16 + j) * 2048);
    }

    STAMP(2);
    unsigned int* tmo = (unsigned int*)(myws + WS_TMO);
    {
        const unsigned int* fl = (const unsigned int*)(myws + WS_FLAGS + (wave * 8 + rb) * 128) + (lane & 3);
        unsigned long long t0 = __builtin_amdgcn_s_memrealtime();
        while (true) {
            unsigned int v = __hip_atomic_load(fl, __ATOMIC_RELAXED, __HIP_MEMORY_SCOPE_SYSTEM);
            if (__all(v >= call)) break;
            if (__hip_atomic_load(tmo, __ATOMIC_RELAXED, __HIP_MEMORY_SCOPE_AGENT) == call
                || __builtin_amdgcn_s_memrealtime() - t0 > DEADLINE_TICKS) {
                if (lane == 0) __hip_atomic_store(tmo, call, __ATOMIC_RELAXED, __HIP_MEMORY_SCOPE_AGENT);
                break;
            }
            __builtin_amdgcn_s_sleep(1);
        }
        STAMP(3);
    }
    STAMP(4);

    f32x16 acc0 = {0}, acc1 = {0};
    {
        const u32x4* ap = (const u32x4*)(myws + WS_STAGE + (call & 1) * SLOT_BYTES + (wave * 8 + rb) * 16 * 1024) + lane;
        u32x4 a[16];
#pragma unroll
        for (int ks = 0; ks < 16; ++ks) {
            unsigned long long* p8 = (unsigned long long*)(ap + ks * 64);
            unsigned long long lo = __hip_atomic_load(p8, __ATOMIC_RELAXED, __HIP_MEMORY_SCOPE_SYSTEM);
            unsigned long long hi = __hip_atomic_load(p8 + 1, __ATOMIC_RELAXED, __HIP_MEMORY_SCOPE_SYSTEM);
            a[ks] = (u32x4){(unsigned)lo, (unsigned)(lo >> 32), (unsigned)hi, (unsigned)(hi >> 32)};
        }
#pragma unroll
        for (int ks = 0; ks < 16; ++ks) {
            u32x4 b0, b1;
#pragma unroll
            for (int j = 0; j < 4; ++j) {
                b0[j] = __builtin_amdgcn_perm(wr[ks][2 * j + 1], wr[ks][2 * j], 0x05040100u);
                b1[j] = __builtin_amdgcn_perm(wr[ks][2 * j + 1], wr[ks][2 * j], 0x07060302u);
            }
            acc0 = __builtin_amdgcn_mfma_f32_32x32x16_bf16(__builtin_bit_cast(bf16x8, a[ks]), __builtin_bit_cast(bf16x8, b0), acc0, 0, 0, 0);
            acc1 = __builtin_amdgcn_mfma_f32_32x32x16_bf16(__builtin_bit_cast(bf16x8, a[ks]), __builtin_bit_cast(bf16x8, b1), acc1, 0, 0, 0);
        }
    }

    asm volatile("" :: "v"(acc0[0]), "v"(acc1[0]));
    STAMP(5);
#pragma unroll
    for (int i = 0; i < 8; ++i) {
        *(f32x2*)&red[(((wave * 2 + 0) * 8 + i) * 64 + lane) * 2] = (f32x2){acc0[2 * i], acc0[2 * i + 1]};
        *(f32x2*)&red[(((wave * 2 + 1) * 8 + i) * 64 + lane) * 2] = (f32x2){acc1[2 * i], acc1[2 * i + 1]};
    }
    __syncthreads();
    const bool bad = __hip_atomic_load(tmo, __ATOMIC_RELAXED, __HIP_MEMORY_SCOPE_AGENT) == call;
    {
        f32x2 s0 = {0, 0}, s1 = {0, 0};
#pragma unroll
        for (int v = 0; v < 8; ++v) {
            s0 += *(const f32x2*)&red[(((v * 2 + 0) * 8 + wave) * 64 + lane) * 2];
            s1 += *(const f32x2*)&red[(((v * 2 + 1) * 8 + wave) * 64 + lane) * 2];
        }
#pragma unroll
        for (int e = 0; e < 2; ++e) {
            const int i = 2 * wave + e;
            const int row = rb * 32 + (i & 3) + 8 * (i >> 2) + 4 * (lane >> 5);
            float v0 = fmaxf(s0[e], 0.f), v1 = fmaxf(s1[e], 0.f);
            unsigned int pk = f2bf_pack(v0, v1);
            if (bad) pk = 0x7fc07fc0u;
            *(unsigned int*)(out + (size_t)row * 2048 + n0 + 2 * (lane & 31)) = pk;
        }
    }
    asm volatile("s_waitcnt vmcnt(0)" ::: "memory");
    STAMP(6);
    if (tid == 0)
        __hip_atomic_fetch_add((unsigned int*)(myws + WS_CTR), 1u, __ATOMIC_RELAXED, __HIP_MEMORY_SCOPE_AGENT);
    (void)sh_call;
}

extern "C" void kernel_launch(int rank, int nranks, ncclComm_t comm,
                              hipStream_t stream,
                              void* const* d_in, const int* in_sizes, int n_in,
                              void* d_out, int out_size,
                              const int* in_dtypes, int out_dtype,
                              void* const* d_out_all,
                              void* const* d_ws_all, size_t ws_bytes) {
    if (nranks != NRANKS || (unsigned)rank >= NRANKS) return;
    hipLaunchKernelGGL(a2a_gemm_kernel, dim3(NWG), dim3(NTHR), 0, stream,
                       (const unsigned short*)d_in[0], (const unsigned short*)d_in[1],
                       (unsigned short*)d_out, d_ws_all, rank);
}
```

```cpp
#include <hip/hip_runtime.h>
#include <rccl/rccl.h>
#include <cstdio>
#include <cstdint>

typedef __attribute__((ext_vector_type(8))) short bf16x8;
typedef __attribute__((ext_vector_type(16))) float f32x16;
typedef __attribute__((ext_vector_type(4))) unsigned int u32x4;
typedef __attribute__((ext_vector_type(2))) float f32x2;

#define NRANKS 8
#define NWG 256
#define NTHR 512
#define DEADLINE_TICKS 50000000ull

#define WS_CTR 0
#define WS_TMO 128
#define WS_FLAGS 4096
#define WS_STAGE (1u << 20)
#define SLOT_BYTES (1u << 20)

__device__ __forceinline__ unsigned int f2bf_pack(float lo, float hi) {
    unsigned short a = __builtin_bit_cast(unsigned short, (__bf16)lo);
    unsigned short b = __builtin_bit_cast(unsigned short, (__bf16)hi);
    return (unsigned int)a | ((unsigned int)b << 16);
}

#define STAMP(i) do { } while (0)

__global__ void __launch_bounds__(NTHR, 2)
a2a_gemm_kernel(const unsigned short* __restrict__ x, const unsigned short* __restrict__ w,
                unsigned short* __restrict__ out, void* const* __restrict__ ws_all, int rank) {
    __shared__ __attribute__((aligned(16))) float red[8 * 2 * 16 * 64];
    __shared__ unsigned int sh_call;
    const int tid = threadIdx.x;
    const int lane = tid & 63;
    const int wave = tid >> 6;
    const int b = blockIdx.x;
    char* myws = (char*)ws_all[rank];
    STAMP(0);

    unsigned int ctr = __hip_atomic_load((unsigned int*)(myws + WS_CTR), __ATOMIC_RELAXED, __HIP_MEMORY_SCOPE_AGENT);

    u32x4 xv;
    const int pd = (rank + (b & 7)) & 7;
    const int prb = b >> 5, pq = (b >> 3) & 3;
    if (tid < 256) {
        const int ks = tid >> 6, l = tid & 63;
        const int row = pd * 256 + prb * 32 + (l & 31);
        const int col = pq * 64 + ks * 16 + (l >> 5) * 8;
        xv = *(const u32x4*)(x + (size_t)row * 256 + col);
    }
    const unsigned int call = ctr / NWG + 1;
    if (tid < 256) {
        char* dws = (char*)ws_all[pd];
        u32x4* dst = (u32x4*)(dws + WS_STAGE + (call & 1) * SLOT_BYTES + ((rank * 8 + prb) * 16 + pq * 4) * 1024) + tid;
        asm volatile("global_store_dwordx4 %0, %1, off sc0 sc1\n\ts_nop 1" :: "v"(dst), "v"(xv) : "memory");
    }
    asm volatile("s_waitcnt vmcnt(0)" ::: "memory");
    STAMP(1);
    __syncthreads();
    if (tid == 0) {
        char* dws = (char*)ws_all[pd];
        __hip_atomic_store((unsigned int*)(dws + WS_FLAGS + (rank * 8 + prb) * 128 + pq * 4), call,
                           __ATOMIC_RELAXED, __HIP_MEMORY_SCOPE_SYSTEM);
    }

    const int idx = b >> 3;
    const int rb = idx >> 2;
    const int cb = (b & 7) + 8 * (idx & 3);
    const int n0 = cb * 64;

    unsigned int wr[16][8];
    {
        const unsigned short* wp = w + (size_t)(wave * 256 + (lane >> 5) * 8) * 2048 + n0 + 2 * (lane & 31);
#pragma unroll
        for (int ks = 0; ks < 16; ++ks)
#pragma unroll
            for (int j = 0; j < 8; ++j)
                wr[ks][j] = *(const unsigned int*)(wp + (size_t)(ks * 16 + j) * 2048);
    }

    STAMP(2);
    unsigned int* tmo = (unsigned int*)(myws + WS_TMO);
    {
        const unsigned int* fl = (const unsigned int*)(myws + WS_FLAGS + (wave * 8 + rb) * 128) + (lane & 3);
        unsigned long long t0 = __builtin_amdgcn_s_memrealtime();
        while (true) {
            unsigned int v = __hip_atomic_load(fl, __ATOMIC_RELAXED, __HIP_MEMORY_SCOPE_SYSTEM);
            if (__all(v >= call)) break;
            if (__hip_atomic_load(tmo, __ATOMIC_RELAXED, __HIP_MEMORY_SCOPE_AGENT) == call
                || __builtin_amdgcn_s_memrealtime() - t0 > DEADLINE_TICKS) {
                if (lane == 0) __hip_atomic_store(tmo, call, __ATOMIC_RELAXED, __HIP_MEMORY_SCOPE_AGENT);
                break;
            }
            __builtin_amdgcn_s_sleep(1);
        }
        STAMP(3);
    }
    STAMP(4);

    f32x16 acc0 = {0}, acc1 = {0};
    {
        const u32x4* ap = (const u32x4*)(myws + WS_STAGE + (call & 1) * SLOT_BYTES + (wave * 8 + rb) * 16 * 1024) + lane;
        u32x4 a[16];
#pragma unroll
        for (int ks = 0; ks < 16; ++ks) {
            unsigned long long* p8 = (unsigned long long*)(ap + ks * 64);
            unsigned long long lo = __hip_atomic_load(p8, __ATOMIC_RELAXED, __HIP_MEMORY_SCOPE_SYSTEM);
            unsigned long long hi = __hip_atomic_load(p8 + 1, __ATOMIC_RELAXED, __HIP_MEMORY_SCOPE_SYSTEM);
            a[ks] = (u32x4){(unsigned)lo, (unsigned)(lo >> 32), (unsigned)hi, (unsigned)(hi >> 32)};
        }
#pragma unroll
        for (int ks = 0; ks < 16; ++ks) {
            u32x4 b0, b1;
#pragma unroll
            for (int j = 0; j < 4; ++j) {
                b0[j] = __builtin_amdgcn_perm(wr[ks][2 * j + 1], wr[ks][2 * j], 0x05040100u);
                b1[j] = __builtin_amdgcn_perm(wr[ks][2 * j + 1], wr[ks][2 * j], 0x07060302u);
            }
            acc0 = __builtin_amdgcn_mfma_f32_32x32x16_bf16(__builtin_bit_cast(bf16x8, a[ks]), __builtin_bit_cast(bf16x8, b0), acc0, 0, 0, 0);
            acc1 = __builtin_amdgcn_mfma_f32_32x32x16_bf16(__builtin_bit_cast(bf16x8, a[ks]), __builtin_bit_cast(bf16x8, b1), acc1, 0, 0, 0);
        }
    }

    asm volatile("" :: "v"(acc0[0]), "v"(acc1[0]));
    STAMP(5);
#pragma unroll
    for (int i = 0; i < 8; ++i) {
        *(f32x2*)&red[(((wave * 2 + 0) * 8 + i) * 64 + lane) * 2] = (f32x2){acc0[2 * i], acc0[2 * i + 1]};
        *(f32x2*)&red[(((wave * 2 + 1) * 8 + i) * 64 + lane) * 2] = (f32x2){acc1[2 * i], acc1[2 * i + 1]};
    }
    __syncthreads();
    const bool bad = __hip_atomic_load(tmo, __ATOMIC_RELAXED, __HIP_MEMORY_SCOPE_AGENT) == call;
    {
        f32x2 s0 = {0, 0}, s1 = {0, 0};
#pragma unroll
        for (int v = 0; v < 8; ++v) {
            s0 += *(const f32x2*)&red[(((v * 2 + 0) * 8 + wave) * 64 + lane) * 2];
            s1 += *(const f32x2*)&red[(((v * 2 + 1) * 8 + wave) * 64 + lane) * 2];
        }
#pragma unroll
        for (int e = 0; e < 2; ++e) {
            const int i = 2 * wave + e;
            const int row = rb * 32 + (i & 3) + 8 * (i >> 2) + 4 * (lane >> 5);
            float v0 = fmaxf(s0[e], 0.f), v1 = fmaxf(s1[e], 0.f);
            unsigned int pk = f2bf_pack(v0, v1);
            if (bad) pk = 0x7fc07fc0u;
            *(unsigned int*)(out + (size_t)row * 2048 + n0 + 2 * (lane & 31)) = pk;
        }
    }
    asm volatile("s_waitcnt vmcnt(0)" ::: "memory");
    STAMP(6);
    if (tid == 0)
        __hip_atomic_fetch_add((unsigned int*)(myws + WS_CTR), 1u, __ATOMIC_RELAXED, __HIP_MEMORY_SCOPE_AGENT);
    (void)sh_call;
}

extern "C" void kernel_launch(int rank, int nranks, ncclComm_t comm,
                              hipStream_t stream,
                              void* const* d_in, const int* in_sizes, int n_in,
                              void* d_out, int out_size,
                              const int* in_dtypes, int out_dtype,
                              void* const* d_out_all,
                              void* const* d_ws_all, size_t ws_bytes) {
    if (nranks != NRANKS || (unsigned)rank >= NRANKS) return;
    hipLaunchKernelGGL(a2a_gemm_kernel, dim3(NWG), dim3(NTHR), 0, stream,
                       (const unsigned short*)d_in[0], (const unsigned short*)d_in[1],
                       (unsigned short*)d_out, d_ws_all, rank);
}
```

```cpp
#include <hip/hip_runtime.h>
#include <rccl/rccl.h>
#include <cstdio>
#include <cstdint>

typedef __attribute__((ext_vector_type(8))) short bf16x8;
typedef __attribute__((ext_vector_type(16))) float f32x16;
typedef __attribute__((ext_vector_type(4))) unsigned int u32x4;
typedef __attribute__((ext_vector_type(2))) float f32x2;
typedef __attribute__((ext_vector_type(4))) float f32x4;
typedef __attribute__((ext_vector_type(2))) unsigned int u32x2;

#define NRANKS 8
#define NWG 256
#define NTHR 512
#define DEADLINE_TICKS 50000000ull

#define WS_CTR 0
#define WS_TMO 128
#define WS_FLAGS 4096
#define WS_STAGE (1u << 20)
#define SLOT_BYTES (1u << 20)

__device__ __forceinline__ unsigned int f2bf_pack(float lo, float hi) {
    unsigned short a = __builtin_bit_cast(unsigned short, (__bf16)lo);
    unsigned short b = __builtin_bit_cast(unsigned short, (__bf16)hi);
    return (unsigned int)a | ((unsigned int)b << 16);
}

#define STAMP(i) do { } while (0)

__global__ void __launch_bounds__(NTHR, 2)
a2a_gemm_kernel(const unsigned short* __restrict__ x, const unsigned short* __restrict__ w,
                unsigned short* __restrict__ out, void* const* __restrict__ ws_all, int rank) {
    __shared__ __attribute__((aligned(16))) float red[8 * 2 * 16 * 64];
    __shared__ unsigned int sh_call;
    const int tid = threadIdx.x;
    const int lane = tid & 63;
    const int wave = tid >> 6;
    const int b = blockIdx.x;
    char* myws = (char*)ws_all[rank];
    STAMP(0);

    unsigned int ctr = __hip_atomic_load((unsigned int*)(myws + WS_CTR), __ATOMIC_RELAXED, __HIP_MEMORY_SCOPE_AGENT);

    u32x4 xv;
    const int pd = (rank + (b & 7)) & 7;
    const int prb = b >> 5, pq = (b >> 3) & 3;
    if (tid < 256) {
        const int ks = tid >> 6, l = tid & 63;
        const int row = pd * 256 + prb * 32 + (ks & 1) * 16 + (l & 15);
        const int col = pq * 64 + (ks >> 1) * 32 + (l >> 4) * 8;
        xv = *(const u32x4*)(x + (size_t)row * 256 + col);
    }
    const unsigned int call = ctr / NWG + 1;
    if (tid < 256) {
        char* dws = (char*)ws_all[pd];
        u32x4* dst = (u32x4*)(dws + WS_STAGE + (call & 1) * SLOT_BYTES + ((rank * 8 + prb) * 16 + pq * 4) * 1024) + tid;
        asm volatile("global_store_dwordx4 %0, %1, off sc0 sc1\n\ts_nop 1" :: "v"(dst), "v"(xv) : "memory");
    }
    const int idx = b >> 3;
    const int rb = idx >> 2;
    const int cb = (b & 7) + 8 * (idx & 3);
    const int n0 = cb * 64;

    u32x2 wr[8][8];
    {
        const unsigned short* wp = w + (size_t)(wave * 256 + (lane >> 4) * 8) * 2048 + n0 + 4 * (lane & 15);
#pragma unroll
        for (int ks = 0; ks < 8; ++ks) {
#pragma unroll
            for (int j = 0; j < 8; ++j)
                wr[ks][j] = *(const u32x2*)(wp + (size_t)(ks * 32 + j) * 2048);
            if (ks == 3 && wave < 4) {
                asm volatile("s_waitcnt vmcnt(32)" ::: "memory");
                STAMP(1);
                if (lane == 0) {
                    char* dws = (char*)ws_all[pd];
                    __hip_atomic_store((unsigned int*)(dws + WS_FLAGS + (rank * 8 + prb) * 128 + (pq * 4 + wave) * 4), call,
                                       __ATOMIC_RELAXED, __HIP_MEMORY_SCOPE_SYSTEM);
                }
            }
        }
    }
    u32x4 bfr[8][4];
#pragma unroll
    for (int ks = 0; ks < 8; ++ks)
#pragma unroll
        for (int t = 0; t < 4; ++t)
#pragma unroll
            for (int j = 0; j < 4; ++j)
                bfr[ks][t][j] = __builtin_amdgcn_perm(wr[ks][2 * j + 1][t >> 1], wr[ks][2 * j][t >> 1], (t & 1) ? 0x07060302u : 0x05040100u);
    STAMP(2);
    unsigned int* tmo = (unsigned int*)(myws + WS_TMO);
    {
        const unsigned int* fl = (const unsigned int*)(myws + WS_FLAGS + (wave * 8 + rb) * 128) + (lane & 15);
        unsigned long long t0 = __builtin_amdgcn_s_memrealtime();
        while (true) {
            unsigned int v = __hip_atomic_load(fl, __ATOMIC_RELAXED, __HIP_MEMORY_SCOPE_SYSTEM);
            if (__all(v >= call)) break;
            if (__hip_atomic_load(tmo, __ATOMIC_RELAXED, __HIP_MEMORY_SCOPE_AGENT) == call
                || __builtin_amdgcn_s_memrealtime() - t0 > DEADLINE_TICKS) {
                if (lane == 0) __hip_atomic_store(tmo, call, __ATOMIC_RELAXED, __HIP_MEMORY_SCOPE_AGENT);
                break;
            }
            __builtin_amdgcn_s_sleep(1);
        }
        STAMP(3);
    }
    STAMP(4);

    f32x4 acc[2][4];
#pragma unroll
    for (int i = 0; i < 2; ++i)
#pragma unroll
        for (int t = 0; t < 4; ++t) acc[i][t] = (f32x4){0.f, 0.f, 0.f, 0.f};
    {
        const u32x4* ap = (const u32x4*)(myws + WS_STAGE + (call & 1) * SLOT_BYTES + (wave * 8 + rb) * 16 * 1024) + lane;
        u32x4 a[16];
#pragma unroll
        for (int f = 0; f < 16; ++f)
            asm volatile("global_load_dwordx4 %0, %1, off sc0 sc1" : "=v"(a[f]) : "v"(ap + f * 64) : "memory");
#pragma unroll
        for (int f = 0; f < 16; ++f) {
            if (f == 0) asm volatile("s_waitcnt vmcnt(8)" : "+v"(a[0]), "+v"(a[1]), "+v"(a[2]), "+v"(a[3]), "+v"(a[4]), "+v"(a[5]), "+v"(a[6]), "+v"(a[7]) :: "memory");
            if (f == 8) asm volatile("s_waitcnt vmcnt(0)" : "+v"(a[8]), "+v"(a[9]), "+v"(a[10]), "+v"(a[11]), "+v"(a[12]), "+v"(a[13]), "+v"(a[14]), "+v"(a[15]) :: "memory");
#pragma unroll
            for (int t = 0; t < 4; ++t)
                acc[f & 1][t] = __builtin_amdgcn_mfma_f32_16x16x32_bf16(__builtin_bit_cast(bf16x8, a[f]), __builtin_bit_cast(bf16x8, bfr[f >> 1][t]), acc[f & 1][t], 0, 0, 0);
        }
    }
    asm volatile("" :: "v"(acc[0][0][0]), "v"(acc[1][3][0]));
    STAMP(5);
    f32x4* red4 = (f32x4*)red;
#pragma unroll
    for (int i = 0; i < 2; ++i)
#pragma unroll
        for (int r = 0; r < 4; ++r)
            red4[((wave * 2 + i) * 4 + r) * 64 + lane] = (f32x4){acc[i][0][r], acc[i][1][r], acc[i][2][r], acc[i][3][r]};
    __syncthreads();
    const bool bad = __hip_atomic_load(tmo, __ATOMIC_RELAXED, __HIP_MEMORY_SCOPE_AGENT) == call;
    {
        f32x4 sm = {0.f, 0.f, 0.f, 0.f};
#pragma unroll
        for (int v = 0; v < 8; ++v) sm += red4[(v * 8 + wave) * 64 + lane];
        const int row = rb * 32 + (wave >> 2) * 16 + (lane >> 4) * 4 + (wave & 3);
        u32x2 pk = {f2bf_pack(fmaxf(sm[0], 0.f), fmaxf(sm[1], 0.f)), f2bf_pack(fmaxf(sm[2], 0.f), fmaxf(sm[3], 0.f))};
        if (bad) pk = (u32x2){0x7fc07fc0u, 0x7fc07fc0u};
        *(u32x2*)(out + (size_t)row * 2048 + n0 + 4 * (lane & 15)) = pk;
    }
    STAMP(6);
    if (tid == 0)
        __hip_atomic_fetch_add((unsigned int*)(myws + WS_CTR), 1u, __ATOMIC_RELAXED, __HIP_MEMORY_SCOPE_AGENT);
    (void)sh_call;
}

extern "C" void kernel_launch(int rank, int nranks, ncclComm_t comm,
                              hipStream_t stream,
                              void* const* d_in, const int* in_sizes, int n_in,
                              void* d_out, int out_size,
                              const int* in_dtypes, int out_dtype,
                              void* const* d_out_all,
                              void* const* d_ws_all, size_t ws_bytes) {
    if (nranks != NRANKS || (unsigned)rank >= NRANKS) return;
    hipLaunchKernelGGL(a2a_gemm_kernel, dim3(NWG), dim3(NTHR), 0, stream,
                       (const unsigned short*)d_in[0], (const unsigned short*)d_in[1],
                       (unsigned short*)d_out, d_ws_all, rank);
}
```

```cpp
#include <hip/hip_runtime.h>
#include <rccl/rccl.h>
#include <cstdio>
#include <cstdint>

typedef __attribute__((ext_vector_type(8))) short bf16x8;
typedef __attribute__((ext_vector_type(16))) float f32x16;
typedef __attribute__((ext_vector_type(4))) unsigned int u32x4;
typedef __attribute__((ext_vector_type(2))) float f32x2;

#define NRANKS 8
#define NWG 256
#define NTHR 512
#define DEADLINE_TICKS 50000000ull

#define WS_CTR 0
#define WS_TMO 128
#define WS_FLAGS 4096
#define WS_STAGE (1u << 20)
#define SLOT_BYTES (1u << 20)

__device__ __forceinline__ unsigned int f2bf_pack(float lo, float hi) {
    unsigned short a = __builtin_bit_cast(unsigned short, (__bf16)lo);
    unsigned short b = __builtin_bit_cast(unsigned short, (__bf16)hi);
    return (unsigned int)a | ((unsigned int)b << 16);
}

#define STAMP(i) do { } while (0)

__global__ void __launch_bounds__(NTHR, 2)
a2a_gemm_kernel(const unsigned short* __restrict__ x, const unsigned short* __restrict__ w,
                unsigned short* __restrict__ out, void* const* __restrict__ ws_all, int rank) {
    __shared__ __attribute__((aligned(16))) float red[8 * 2 * 16 * 64];
    __shared__ unsigned int sh_call;
    const int tid = threadIdx.x;
    const int lane = tid & 63;
    const int wave = tid >> 6;
    const int b = blockIdx.x;
    char* myws = (char*)ws_all[rank];
    STAMP(0);

    unsigned int ctr = __hip_atomic_load((unsigned int*)(myws + WS_CTR), __ATOMIC_RELAXED, __HIP_MEMORY_SCOPE_AGENT);

    u32x4 xv;
    const int pd = (rank + (b & 7)) & 7;
    const int prb = b >> 5, pq = (b >> 3) & 3;
    if (tid < 256) {
        const int ks = tid >> 6, l = tid & 63;
        const int row = pd * 256 + prb * 32 + (l & 31);
        const int col = pq * 64 + ks * 16 + (l >> 5) * 8;
        xv = *(const u32x4*)(x + (size_t)row * 256 + col);
    }
    const unsigned int call = ctr / NWG + 1;
    if (tid < 256) {
        char* dws = (char*)ws_all[pd];
        u32x4* dst = (u32x4*)(dws + WS_STAGE + (call & 1) * SLOT_BYTES + ((rank * 8 + prb) * 16 + pq * 4) * 1024) + tid;
        asm volatile("global_store_dwordx4 %0, %1, off sc0 sc1\n\ts_nop 1" :: "v"(dst), "v"(xv) : "memory");
    }
    const int idx = b >> 3;
    const int rb = idx >> 2;
    const int cb = (b & 7) + 8 * (idx & 3);
    const int n0 = cb * 64;

    unsigned int wr[16][8];
    {
        const unsigned short* wp = w + (size_t)(wave * 256 + (lane >> 5) * 8) * 2048 + n0 + 2 * (lane & 31);
#pragma unroll
        for (int ks = 0; ks < 16; ++ks) {
#pragma unroll
            for (int j = 0; j < 8; ++j)
                wr[ks][j] = *(const unsigned int*)(wp + (size_t)(ks * 16 + j) * 2048);
            if (ks == 3 && wave < 4) {
                asm volatile("s_waitcnt vmcnt(32)" ::: "memory");
                STAMP(1);
                if (lane == 0) {
                    char* dws = (char*)ws_all[pd];
                    __hip_atomic_store((unsigned int*)(dws + WS_FLAGS + (rank * 8 + prb) * 128 + (pq * 4 + wave) * 4), call,
                                       __ATOMIC_RELAXED, __HIP_MEMORY_SCOPE_SYSTEM);
                }
            }
        }
    }
    u32x4 bf0[16], bf1[16];
#pragma unroll
    for (int ks = 0; ks < 16; ++ks)
#pragma unroll
        for (int j = 0; j < 4; ++j) {
            bf0[ks][j] = __builtin_amdgcn_perm(wr[ks][2 * j + 1], wr[ks][2 * j], 0x05040100u);
            bf1[ks][j] = __builtin_amdgcn_perm(wr[ks][2 * j + 1], wr[ks][2 * j], 0x07060302u);
        }

    STAMP(2);
    unsigned int* tmo = (unsigned int*)(myws + WS_TMO);
    {
        const unsigned int* fl = (const unsigned int*)(myws + WS_FLAGS + (wave * 8 + rb) * 128) + (lane & 15);
        unsigned long long t0 = __builtin_amdgcn_s_memrealtime();
        while (true) {
            unsigned int v = __hip_atomic_load(fl, __ATOMIC_RELAXED, __HIP_MEMORY_SCOPE_SYSTEM);
            if (__all(v >= call)) break;
            if (__hip_atomic_load(tmo, __ATOMIC_RELAXED, __HIP_MEMORY_SCOPE_AGENT) == call
                || __builtin_amdgcn_s_memrealtime() - t0 > DEADLINE_TICKS) {
                if (lane == 0) __hip_atomic_store(tmo, call, __ATOMIC_RELAXED, __HIP_MEMORY_SCOPE_AGENT);
                break;
            }
            __builtin_amdgcn_s_sleep(1);
        }
        STAMP(3);
    }
    STAMP(4);

    f32x16 acc0 = {0}, acc1 = {0};
    {
        const u32x4* ap = (const u32x4*)(myws + WS_STAGE + (call & 1) * SLOT_BYTES + (wave * 8 + rb) * 16 * 1024) + lane;
        u32x4 a[16];
#pragma unroll
        for (int ks = 0; ks < 16; ++ks)
            asm volatile("global_load_dwordx4 %0, %1, off sc0 sc1" : "=v"(a[ks]) : "v"(ap + ks * 64) : "memory");
#pragma unroll
        for (int ks = 0; ks < 16; ++ks) {
            if (ks == 0) asm volatile("s_waitcnt vmcnt(8)" : "+v"(a[0]), "+v"(a[1]), "+v"(a[2]), "+v"(a[3]), "+v"(a[4]), "+v"(a[5]), "+v"(a[6]), "+v"(a[7]) :: "memory");
            if (ks == 8) asm volatile("s_waitcnt vmcnt(0)" : "+v"(a[8]), "+v"(a[9]), "+v"(a[10]), "+v"(a[11]), "+v"(a[12]), "+v"(a[13]), "+v"(a[14]), "+v"(a[15]) :: "memory");
            acc0 = __builtin_amdgcn_mfma_f32_32x32x16_bf16(__builtin_bit_cast(bf16x8, a[ks]), __builtin_bit_cast(bf16x8, bf0[ks]), acc0, 0, 0, 0);
            acc1 = __builtin_amdgcn_mfma_f32_32x32x16_bf16(__builtin_bit_cast(bf16x8, a[ks]), __builtin_bit_cast(bf16x8, bf1[ks]), acc1, 0, 0, 0);
        }
    }
    asm volatile("" :: "v"(acc0[0]), "v"(acc1[0]));
    STAMP(5);
#pragma unroll
    for (int i = 0; i < 8; ++i) {
        *(f32x2*)&red[(((wave * 2 + 0) * 8 + i) * 64 + lane) * 2] = (f32x2){acc0[2 * i], acc0[2 * i + 1]};
        *(f32x2*)&red[(((wave * 2 + 1) * 8 + i) * 64 + lane) * 2] = (f32x2){acc1[2 * i], acc1[2 * i + 1]};
    }
    __syncthreads();
    const bool bad = __hip_atomic_load(tmo, __ATOMIC_RELAXED, __HIP_MEMORY_SCOPE_AGENT) == call;
    {
        f32x2 s0 = {0, 0}, s1 = {0, 0};
#pragma unroll
        for (int v = 0; v < 8; ++v) {
            s0 += *(const f32x2*)&red[(((v * 2 + 0) * 8 + wave) * 64 + lane) * 2];
            s1 += *(const f32x2*)&red[(((v * 2 + 1) * 8 + wave) * 64 + lane) * 2];
        }
#pragma unroll
        for (int e = 0; e < 2; ++e) {
            const int i = 2 * wave + e;
            const int row = rb * 32 + (i & 3) + 8 * (i >> 2) + 4 * (lane >> 5);
            float v0 = fmaxf(s0[e], 0.f), v1 = fmaxf(s1[e], 0.f);
            unsigned int pk = f2bf_pack(v0, v1);
            if (bad) pk = 0x7fc07fc0u;
            *(unsigned int*)(out + (size_t)row * 2048 + n0 + 2 * (lane & 31)) = pk;
        }
    }
    asm volatile("s_waitcnt vmcnt(0)" ::: "memory");
    STAMP(6);
    if (tid == 0)
        __hip_atomic_fetch_add((unsigned int*)(myws + WS_CTR), 1u, __ATOMIC_RELAXED, __HIP_MEMORY_SCOPE_AGENT);
    (void)sh_call;
}

extern "C" void kernel_launch(int rank, int nranks, ncclComm_t comm,
                              hipStream_t stream,
                              void* const* d_in, const int* in_sizes, int n_in,
                              void* d_out, int out_size,
                              const int* in_dtypes, int out_dtype,
                              void* const* d_out_all,
                              void* const* d_ws_all, size_t ws_bytes) {
    if (nranks != NRANKS || (unsigned)rank >= NRANKS) return;
    hipLaunchKernelGGL(a2a_gemm_kernel, dim3(NWG), dim3(NTHR), 0, stream,
                       (const unsigned short*)d_in[0], (const unsigned short*)d_in[1],
                       (unsigned short*)d_out, d_ws_all, rank);
}
```
